```python
import numpy as np

kernel_name = "gpu_ag_gemm_m4096_k4096_n16384_f32_relu"


ABSMAX_RTOL = 0.001

COMPUTE_DTYPE = "float32"
EPILOGUE = "relu"
IN_QUANT = None
OUT_QUANT = None
IN_DTYPES = {"x": "float32", "w_mat": "float32"}
WORLD_SIZE = 8

def setup_inputs():

    x = np.random.randn(4096, 4096).astype(np.float32)
    w_mat = (np.random.randn(4096, 16384) / np.sqrt(4096)).astype(np.float32)
    return {"x": x, "w_mat": w_mat}

def _epilogue(y):
    return np.maximum(y, 0.0).astype(np.float32)

def reference(x, w_mat):
    y = (x @ w_mat).astype(np.float32)
    return _epilogue(y)
```

```cpp
#include <hip/hip_runtime.h>
#include <rccl/rccl.h>
#include <cstdio>

#define HIP_CHECK(expr) do { hipError_t _e = (expr); if (_e != hipSuccess) { \
    fprintf(stderr, "HIP error %s at %s:%d\n", hipGetErrorString(_e), __FILE__, __LINE__); \
    return; } } while (0)

namespace {

typedef _Float16 h8 __attribute__((ext_vector_type(8)));
typedef float f4 __attribute__((ext_vector_type(4)));
typedef unsigned u4 __attribute__((ext_vector_type(4)));

constexpr int NR = 8;
constexpr int MS = 512;
constexpr int KD = 4096;
constexpr int NS = 2048;
constexpr int NSTG = KD / 64;
constexpr int NCH = 32;
constexpr int NPUSH = 48, NPW = 48, NTN = 13, NGEMM = 16 * NTN;
constexpr unsigned XCONTRIB = 128;
constexpr unsigned WCONTRIB = 16;

constexpr size_t OFF_WGCNT = 0;
constexpr size_t OFF_TMO = 4096;
constexpr size_t OFF_XCNT = 8192;
constexpr size_t OFF_WCNT = OFF_XCNT + 8 * 32 * 128;
constexpr size_t OFF_XF = 1ull << 20;
constexpr size_t XF_RANK = 4ull << 20;
constexpr size_t XF_PAR = 32ull << 20;
constexpr size_t OFF_WF = OFF_XF + 2 * XF_PAR;
constexpr size_t WF_KF = 130 * 1024;
constexpr size_t WS_NEED = OFF_WF + 128 * WF_KF;

constexpr int STAGE_B = 52 * 1024;
constexpr int LDS_BYTES = 3 * STAGE_B;

constexpr unsigned long long DEADLINE = 50000000ull;

struct Params {
  const float* x;
  const float* w;
  float* out;
  void* const* ws_all;
  int rank;
};

__device__ __forceinline__ void st16_sys(void* p, u4 v) {
  asm volatile("global_store_dwordx4 %0, %1, off sc0 sc1\n\ts_nop 1" :: "v"(p), "v"(v) : "memory");
}

__device__ __forceinline__ u4 cvt8(f4 a, f4 b) {
  h8 h;
  h[0] = (_Float16)a[0]; h[1] = (_Float16)a[1]; h[2] = (_Float16)a[2]; h[3] = (_Float16)a[3];
  h[4] = (_Float16)b[0]; h[5] = (_Float16)b[1]; h[6] = (_Float16)b[2]; h[7] = (_Float16)b[3];
  return __builtin_bit_cast(u4, h);
}

#define STAMP(i) do {} while (0)
#define WAIT_VM(n) asm volatile("s_waitcnt vmcnt(" #n ")" ::: "memory")
#define WAIT_LGKM0() asm volatile("s_waitcnt lgkmcnt(0)" ::: "memory")
#define BAR() __builtin_amdgcn_s_barrier()
#define GLDS(g, l) __builtin_amdgcn_global_load_lds((const unsigned*)(g), (unsigned*)(l), 16, 0, 0)

__device__ __forceinline__ void spin_wait(unsigned* flag, unsigned target, unsigned* tmo, unsigned call) {
  unsigned long long t0 = __builtin_amdgcn_s_memrealtime();
  while (__hip_atomic_load(flag, __ATOMIC_RELAXED, __HIP_MEMORY_SCOPE_SYSTEM) < target) {
    if (__hip_atomic_load(tmo, __ATOMIC_RELAXED, __HIP_MEMORY_SCOPE_AGENT) == call ||
        __builtin_amdgcn_s_memrealtime() - t0 > DEADLINE) {
      __hip_atomic_store(tmo, call, __ATOMIC_RELAXED, __HIP_MEMORY_SCOPE_AGENT);
      break;
    }
    __builtin_amdgcn_s_sleep(2);
  }
}

__global__ void __launch_bounds__(512, 2) ag_gemm_kernel(Params p) {
  extern __shared__ __attribute__((aligned(16))) char smem[];
  const int tid = threadIdx.x;
  const int lane = tid & 63;
  const int wv = __builtin_amdgcn_readfirstlane(tid >> 6);
  const int bid = blockIdx.x;
  char* ws = (char*)p.ws_all[p.rank];

  unsigned* mycnt = (unsigned*)(ws + OFF_WGCNT) + bid;
  const unsigned call = __builtin_amdgcn_readfirstlane(
      __hip_atomic_load(mycnt, __ATOMIC_RELAXED, __HIP_MEMORY_SCOPE_AGENT)) + 1u;
  const unsigned par = call & 1u;
  unsigned* tmo = (unsigned*)(ws + OFF_TMO);
  STAMP(0);

  if (bid < NPUSH) {
    const int role = ((bid & 7) + (bid >> 3)) & 7;
    const int wi = (bid >> 3) * 8 + wv;
    const int dest = (p.rank + 1 + role) & 7;
    char* dws = (char*)p.ws_all[dest];
    char* dst = dws + OFF_XF + par * XF_PAR + (size_t)p.rank * XF_RANK + lane * 16;
    unsigned* cnt = (unsigned*)(dws + OFF_XCNT + (size_t)p.rank * 32 * 128);
    const char* src = (const char*)(p.x + (size_t)(lane & 15) * KD + (lane >> 4) * 8);
    char* ring = smem + wv * 4096 + lane * 16;
    const int nf = (4096 - wi + NPW - 1) / NPW;
#define SRCOFF(F) ((size_t)((F) & 31) * (16 * KD * 4) + (size_t)((F) >> 5) * 128)
    GLDS(src + SRCOFF(wi), ring); GLDS(src + SRCOFF(wi) + 16, ring + 1024);
    int done = 0;
    bool sig_prev = false;
    for (int j = 0; j < nf; ++j) {
      const int f = wi + j * NPW, sl = j & 1;
      if (j + 1 < nf) {
        const size_t so = SRCOFF(f + NPW);
        GLDS(src + so, ring + (sl ^ 1) * 2048); GLDS(src + so + 16, ring + (sl ^ 1) * 2048 + 1024);
        if (j == 0) { WAIT_VM(2); } else if (sig_prev) { WAIT_VM(4); } else { WAIT_VM(3); }
      } else {
        WAIT_VM(1);
      }
      sig_prev = false;
      if (j >= 2) {
        ++done;
        const int c2 = (f - 2 * NPW) >> 7;
        if (((f - NPW) >> 7) != c2) {
          if (lane == 0) __hip_atomic_fetch_add(cnt + c2 * 32, (unsigned)done, __ATOMIC_RELAXED, __HIP_MEMORY_SCOPE_SYSTEM);
          done = 0; sig_prev = true;
        }
      }
      f4 x0 = *(const f4*)(ring + sl * 2048), x1 = *(const f4*)(ring + sl * 2048 + 1024);
      st16_sys(dst + (size_t)f * 1024, cvt8(x0, x1));
    }
    WAIT_VM(0);
    {
      const int fl = wi + (nf - 1) * NPW, c1 = fl >> 7, c2 = (fl - NPW) >> 7;
      if (lane == 0) {
        if (c1 == c2) {
          __hip_atomic_fetch_add(cnt + c1 * 32, (unsigned)(done + 2), __ATOMIC_RELAXED, __HIP_MEMORY_SCOPE_SYSTEM);
        } else {
          __hip_atomic_fetch_add(cnt + c2 * 32, (unsigned)(done + 1), __ATOMIC_RELAXED, __HIP_MEMORY_SCOPE_SYSTEM);
          __hip_atomic_fetch_add(cnt + c1 * 32, 1u, __ATOMIC_RELAXED, __HIP_MEMORY_SCOPE_SYSTEM);
        }
      }
    }
    STAMP(32);
    if (tid == 0) __hip_atomic_store(mycnt, call, __ATOMIC_RELAXED, __HIP_MEMORY_SCOPE_AGENT);
    return;
  }

  const int g = bid - NPUSH;
  const int src = g & 7, idx = g >> 3;
  const int half = idx & 1, nt = idx >> 1;
  const int mt = src * 2 + half;
  const bool has5 = nt < NTN - 1;
  const int wcols = has5 ? 80 : 64;
  const int wm = wv >> 1, wn = wv & 1;
  unsigned xtarget = call * XCONTRIB;
  const unsigned wtarget = call * WCONTRIB;
  unsigned* xcnt = (unsigned*)(ws + OFF_XCNT + (size_t)src * 32 * 128);
  unsigned* wcnt = (unsigned*)(ws + OFF_WCNT) + nt * 32;

  {
    const int kf = mt * 8 + wv;
    const float* wrow = p.w + (size_t)(kf * 32 + (lane >> 4) * 8) * NS + nt * 160;
    char* wdst = ws + OFF_WF + (size_t)kf * WF_KF + (size_t)nt * 10240 + lane * 16;
    f4 a[2][8]; float e[2][8];
    #pragma unroll
    for (int h = 0; h < 2; ++h)
      #pragma unroll
      for (int q = 0; q < 8; ++q) a[h][q] = *(const f4*)(wrow + (size_t)q * NS + h * wcols + (lane & 15) * 4);
    if (has5) {
      #pragma unroll
      for (int h = 0; h < 2; ++h)
        #pragma unroll
        for (int q = 0; q < 8; ++q) e[h][q] = wrow[(size_t)q * NS + h * 80 + 64 + (lane & 15)];
    }
    #pragma unroll
    for (int h = 0; h < 2; ++h) {
      #pragma unroll
      for (int u = 0; u < 4; ++u) {
        h8 v;
        #pragma unroll
        for (int q = 0; q < 8; ++q) v[q] = (_Float16)a[h][q][u];
        st16_sys(wdst + (h * 5 + u) * 1024, __builtin_bit_cast(u4, v));
      }
      if (has5) {
        h8 v;
        #pragma unroll
        for (int q = 0; q < 8; ++q) v[q] = (_Float16)e[h][q];
        st16_sys(wdst + (h * 5 + 4) * 1024, __builtin_bit_cast(u4, v));
      }
    }
    WAIT_VM(0);
    BAR();
    if (tid == 0) __hip_atomic_fetch_add(wcnt, 1u, __ATOMIC_RELAXED, __HIP_MEMORY_SCOPE_AGENT);
  }

  const char* gA = ws + OFF_XF + par * XF_PAR + (size_t)src * XF_RANK +
                   (size_t)((wv >> 2) * 32 + half * 16 + (wv & 3) * 4) * 1024 + lane * 16;
  char* ldA = smem + ((wv >> 2) * 16 + (wv & 3) * 4) * 1024 + lane * 16;
  const int pb2 = wv < 4 ? wv + 16 : wv + 8;
  const char* gB0 = ws + OFF_WF + (size_t)nt * 10240 + lane * 16;
  const size_t ob0 = (size_t)(wv / 10) * WF_KF + (wv % 10) * 1024;
  const size_t ob1 = (size_t)((wv + 8) / 10) * WF_KF + ((wv + 8) % 10) * 1024;
  const size_t ob2 = (size_t)(pb2 / 10) * WF_KF + (pb2 % 10) * 1024;
  char* ldB = smem + 32768 + lane * 16;
  const char* rdA = smem + (wm * 4) * 1024 + lane * 16;
  const char* rdB = smem + 32768 + (wn * 5) * 1024 + lane * 16;

  f4 acc[4][5];
  #pragma unroll
  for (int m = 0; m < 4; ++m)
    #pragma unroll
    for (int n = 0; n < 5; ++n) acc[m][n] = f4{0.f, 0.f, 0.f, 0.f};
  h8 fa[2][4], fb[2][5];
  unsigned fx = 0;

#define ISSUE(S, B) do { \
    _Pragma("unroll") for (int i_ = 0; i_ < 4; ++i_) \
      GLDS(gA + (size_t)(S) * 65536 + i_ * 1024, ldA + (B) * STAGE_B + i_ * 1024); \
    GLDS(gB0 + (size_t)(S) * (2 * WF_KF) + ob0, ldB + (B) * STAGE_B + wv * 1024); \
    GLDS(gB0 + (size_t)(S) * (2 * WF_KF) + ob1, ldB + (B) * STAGE_B + (wv + 8) * 1024); \
    GLDS(gB0 + (size_t)(S) * (2 * WF_KF) + ob2, ldB + (B) * STAGE_B + pb2 * 1024); \
  } while (0)
#define READ(SET, B, KH) do { \
    _Pragma("unroll") for (int n_ = 0; n_ < 5; ++n_) fb[SET][n_] = *(const h8*)(rdB + (B) * STAGE_B + (KH) * 10240 + n_ * 1024); \
    _Pragma("unroll") for (int m_ = 0; m_ < 4; ++m_) fa[SET][m_] = *(const h8*)(rdA + (B) * STAGE_B + (KH) * 16384 + m_ * 1024); \
  } while (0)
#define MMA(SET) do { \
    _Pragma("unroll") for (int m_ = 0; m_ < 4; ++m_) \
      _Pragma("unroll") for (int n_ = 0; n_ < 5; ++n_) \
        acc[m_][n_] = __builtin_amdgcn_mfma_f32_16x16x32_f16(fa[SET][m_], fb[SET][n_], acc[m_][n_], 0, 0, 0); \
  } while (0)
#define FLAGLOAD(Q) asm volatile("global_load_dword %0, %1, off sc0 sc1" : "=v"(fx) : "v"(xcnt + (Q) * 32) : "memory")

  if (wv == 0) {
    spin_wait(wcnt, wtarget, tmo, call);
    spin_wait(xcnt, xtarget, tmo, call);
    spin_wait(xcnt + 32, xtarget, tmo, call);
    __builtin_amdgcn_fence(__ATOMIC_ACQUIRE, "");
    WAIT_VM(0);
  }
  BAR();
  STAMP(1);
  ISSUE(0, 0); ISSUE(1, 1);
  if (wv == 0) FLAGLOAD(2);
  if (wv < 4) ISSUE(2, 2);
  if (wv == 0) { WAIT_VM(15); } else if (wv < 4) { WAIT_VM(14); } else { WAIT_VM(7); }
  BAR();
  READ(0, 0, 0);

#define STEPA(T, P) do { \
    const int t_ = (T); \
    READ(1, (P), 1); \
    MMA(0); \
    if (t_ + 2 >= NSTG) { WAIT_VM(0); } \
    else if (wv == 0 && (t_ & 1) == 0 && t_ + 3 < NSTG - 1) { WAIT_VM(8); } \
    else { WAIT_VM(7); } \
    if (wv == 0 && (t_ & 1) == 1 && t_ + 3 < NSTG) { \
      asm volatile("" : "+v"(fx)); \
      if (__builtin_amdgcn_readfirstlane(fx) < xtarget) spin_wait(xcnt + ((t_ + 3) >> 1) * 32, xtarget, tmo, call); \
      STAMP(1 + ((t_ + 3) >> 1)); \
    } \
    WAIT_LGKM0(); \
    BAR(); \
    if (t_ + 3 < NSTG) ISSUE(t_ + 3, (P)); \
    if (wv == 0 && (t_ & 1) == 1 && t_ + 5 < NSTG) FLAGLOAD((t_ + 5) >> 1); \
    if (t_ + 1 < NSTG) READ(0, ((P) + 1) % 3, 0); \
    MMA(1); \
  } while (0)
#define STEPB(T, P) do { \
    const int t_ = (T); \
    if (t_ + 2 < NSTG) ISSUE(t_ + 2, ((P) + 2) % 3); \
    READ(1, (P), 1); \
    MMA(0); \
    if (t_ + 2 >= NSTG) { WAIT_VM(0); } else { WAIT_VM(7); } \
    WAIT_LGKM0(); \
    BAR(); \
    if (t_ + 1 < NSTG) READ(0, ((P) + 1) % 3, 0); \
    MMA(1); \
  } while (0)

  if (wv < 4) {
    for (int t = 0; t < 63; t += 3) { STEPA(t, 0); STEPA(t + 1, 1); STEPA(t + 2, 2); }
    STEPA(63, 0);
  } else {
    for (int t = 0; t < 63; t += 3) { STEPB(t, 0); STEPB(t + 1, 1); STEPB(t + 2, 2); }
    STEPB(63, 0);
  }

  STAMP(33);
  const bool bad = __hip_atomic_load(tmo, __ATOMIC_RELAXED, __HIP_MEMORY_SCOPE_AGENT) == call;
  const float qnan = __builtin_nanf("");
  float* o = p.out + (size_t)(mt * 256 + wm * 64 + (lane >> 4) * 4) * NS + nt * 160 + wn * wcols;
  #pragma unroll
  for (int m = 0; m < 4; ++m)
    #pragma unroll
    for (int r = 0; r < 4; ++r) {
      f4 v;
      #pragma unroll
      for (int n = 0; n < 4; ++n) v[n] = bad ? qnan : fmaxf(acc[m][n][r], 0.f);
      *(f4*)(o + (size_t)(m * 16 + r) * NS + (lane & 15) * 4) = v;
      if (has5) o[(size_t)(m * 16 + r) * NS + 64 + (lane & 15)] = bad ? qnan : fmaxf(acc[m][4][r], 0.f);
    }
  STAMP(34);
  if (tid == 0) __hip_atomic_store(mycnt, call, __ATOMIC_RELAXED, __HIP_MEMORY_SCOPE_AGENT);
}

}

extern "C" void kernel_launch(int rank, int nranks, ncclComm_t comm,
                              hipStream_t stream,
                              void* const* d_in, const int* in_sizes, int n_in,
                              void* d_out, int out_size,
                              const int* in_dtypes, int out_dtype,
                              void* const* d_out_all,
                              void* const* d_ws_all, size_t ws_bytes) {
  static bool init = false;
  if (!init) {
    if (nranks != NR || ws_bytes < WS_NEED || n_in != 2 || in_sizes[0] != MS * KD ||
        in_sizes[1] != KD * NS || out_size != NR * MS * NS) {
      fprintf(stderr, "unexpected problem shape\n");
      return;
    }
    HIP_CHECK(hipFuncSetAttribute((const void*)ag_gemm_kernel,
                                  hipFuncAttributeMaxDynamicSharedMemorySize, LDS_BYTES));
    init = true;
  }
  Params p;
  p.x = (const float*)d_in[0];
  p.w = (const float*)d_in[1];
  p.out = (float*)d_out;
  p.ws_all = d_ws_all;
  p.rank = rank;
  hipLaunchKernelGGL(ag_gemm_kernel, dim3(NPUSH + NGEMM), dim3(512), LDS_BYTES, stream, p);
}
```

```cpp
#include <hip/hip_runtime.h>
#include <rccl/rccl.h>
#include <cstdio>

#define HIP_CHECK(expr) do { hipError_t _e = (expr); if (_e != hipSuccess) { \
    fprintf(stderr, "HIP error %s at %s:%d\n", hipGetErrorString(_e), __FILE__, __LINE__); \
    return; } } while (0)

namespace {

typedef _Float16 h8 __attribute__((ext_vector_type(8)));
typedef float f4 __attribute__((ext_vector_type(4)));
typedef unsigned u4 __attribute__((ext_vector_type(4)));

constexpr int NR = 8;
constexpr int MS = 512;
constexpr int KD = 4096;
constexpr int NS = 2048;
constexpr int NSTG = KD / 64;
constexpr int NCH = 32;
constexpr int NPUSH = 48, NPW = 48, NTN = 13, NGEMM = 16 * NTN;
constexpr unsigned XCONTRIB = 128;
constexpr unsigned WCONTRIB = 16;

constexpr size_t OFF_WGCNT = 0;
constexpr size_t OFF_TMO = 4096;
constexpr size_t OFF_XCNT = 8192;
constexpr size_t OFF_WCNT = OFF_XCNT + 8 * 32 * 128;
constexpr size_t OFF_XF = 1ull << 20;
constexpr size_t XF_RANK = 4ull << 20;
constexpr size_t XF_PAR = 32ull << 20;
constexpr size_t OFF_WF = OFF_XF + 2 * XF_PAR;
constexpr size_t WF_KF = 130 * 1024;
constexpr size_t WS_NEED = OFF_WF + 128 * WF_KF;

constexpr int STAGE_B = 52 * 1024;
constexpr int LDS_BYTES = 3 * STAGE_B;

constexpr unsigned long long DEADLINE = 50000000ull;

struct Params {
  const float* x;
  const float* w;
  float* out;
  void* const* ws_all;
  int rank;
};

__device__ __forceinline__ void st16_sys(void* p, u4 v) {
  asm volatile("global_store_dwordx4 %0, %1, off sc0 sc1\n\ts_nop 1" :: "v"(p), "v"(v) : "memory");
}

__device__ __forceinline__ u4 cvt8(f4 a, f4 b) {
  h8 h;
  h[0] = (_Float16)a[0]; h[1] = (_Float16)a[1]; h[2] = (_Float16)a[2]; h[3] = (_Float16)a[3];
  h[4] = (_Float16)b[0]; h[5] = (_Float16)b[1]; h[6] = (_Float16)b[2]; h[7] = (_Float16)b[3];
  return __builtin_bit_cast(u4, h);
}

#define STAMP(i) do {} while (0)
#define WAIT_VM(n) asm volatile("s_waitcnt vmcnt(" #n ")" ::: "memory")
#define WAIT_LGKM0() asm volatile("s_waitcnt lgkmcnt(0)" ::: "memory")
#define BAR() __builtin_amdgcn_s_barrier()
#define GLDS(g, l) __builtin_amdgcn_global_load_lds((const unsigned*)(g), (unsigned*)(l), 16, 0, 0)

#define PD 1
__device__ __forceinline__ void wait_vm_n(int n) {
  switch (n) {
    case 0: WAIT_VM(0); break;   case 1: WAIT_VM(1); break;   case 2: WAIT_VM(2); break;
    case 3: WAIT_VM(3); break;   case 4: WAIT_VM(4); break;   case 5: WAIT_VM(5); break;
    case 6: WAIT_VM(6); break;   case 7: WAIT_VM(7); break;   case 8: WAIT_VM(8); break;
    case 9: WAIT_VM(9); break;   case 10: WAIT_VM(10); break; case 11: WAIT_VM(11); break;
    default: WAIT_VM(12); break;
  }
}

__device__ __forceinline__ void spin_wait(unsigned* flag, unsigned target, unsigned* tmo, unsigned call) {
  unsigned long long t0 = __builtin_amdgcn_s_memrealtime();
  while (__hip_atomic_load(flag, __ATOMIC_RELAXED, __HIP_MEMORY_SCOPE_SYSTEM) < target) {
    if (__hip_atomic_load(tmo, __ATOMIC_RELAXED, __HIP_MEMORY_SCOPE_AGENT) == call ||
        __builtin_amdgcn_s_memrealtime() - t0 > DEADLINE) {
      __hip_atomic_store(tmo, call, __ATOMIC_RELAXED, __HIP_MEMORY_SCOPE_AGENT);
      break;
    }
    __builtin_amdgcn_s_sleep(2);
  }
}

__global__ void __launch_bounds__(512, 2) ag_gemm_kernel(Params p) {
  extern __shared__ __attribute__((aligned(16))) char smem[];
  const int tid = threadIdx.x;
  const int lane = tid & 63;
  const int wv = __builtin_amdgcn_readfirstlane(tid >> 6);
  const int bid = blockIdx.x;
  char* ws = (char*)p.ws_all[p.rank];

#define SRCOFF(F) ((size_t)((F) & 31) * (16 * KD * 4) + (size_t)((F) >> 5) * 128)
  if (bid < NPUSH) {
    const char* src0 = (const char*)(p.x + (size_t)(lane & 15) * KD + (lane >> 4) * 8) + SRCOFF((bid >> 3) * 8 + wv);
    char* ring0 = smem + wv * 8192 + lane * 16;
    GLDS(src0, ring0); GLDS(src0 + 16, ring0 + 1024);
  }
  unsigned* mycnt = (unsigned*)(ws + OFF_WGCNT) + bid;
  const unsigned call = __builtin_amdgcn_readfirstlane(
      __hip_atomic_load(mycnt, __ATOMIC_RELAXED, __HIP_MEMORY_SCOPE_AGENT)) + 1u;
  const unsigned par = call & 1u;
  unsigned* tmo = (unsigned*)(ws + OFF_TMO);
  STAMP(0);

  if (bid < NPUSH) {
    const int role = ((bid & 7) + (bid >> 3)) & 7;
    const int wi = (bid >> 3) * 8 + wv;
    const int dest = (p.rank + 1 + role) & 7;
    char* dws = (char*)p.ws_all[dest];
    char* dst = dws + OFF_XF + par * XF_PAR + (size_t)p.rank * XF_RANK + lane * 16;
    unsigned* cnt = (unsigned*)(dws + OFF_XCNT + (size_t)p.rank * 32 * 128);
    const char* src = (const char*)(p.x + (size_t)(lane & 15) * KD + (lane >> 4) * 8);
    char* ring = smem + wv * 8192 + lane * 16;
    const int nf = (4096 - wi + NPW - 1) / NPW;
    #pragma unroll
    for (int i = 1; i < PD; ++i) { const size_t so = SRCOFF(wi + i * NPW); GLDS(src + so, ring + i * 2048); GLDS(src + so + 16, ring + i * 2048 + 1024); }
    int done = 0;
    unsigned hist = 0;
    int sl = 0, psl = PD;
    for (int j = 0; j < nf; ++j) {
      const int f = wi + j * NPW;
      if (j + PD < nf) {
        const size_t so = SRCOFF(f + PD * NPW);
        GLDS(src + so, ring + psl * 2048); GLDS(src + so + 16, ring + psl * 2048 + 1024);
      }
      const int ng = (nf - 1 - j) < PD ? (nf - 1 - j) : PD;
      const int ns = j < PD ? j : PD;
      wait_vm_n(ns + 2 * ng + __builtin_popcount(hist & ((1u << PD) - 1)));
      hist <<= 1;
      if (j >= PD + 1) {
        ++done;
        const int c2 = (f - (PD + 1) * NPW) >> 7;
        if (((f - PD * NPW) >> 7) != c2) {
          if (lane == 0) __hip_atomic_fetch_add(cnt + c2 * 32, (unsigned)done, __ATOMIC_RELAXED, __HIP_MEMORY_SCOPE_SYSTEM);
          done = 0; hist |= 1;
        }
      }
      f4 x0 = *(const f4*)(ring + sl * 2048), x1 = *(const f4*)(ring + sl * 2048 + 1024);
      st16_sys(dst + (size_t)f * 1024, cvt8(x0, x1));
      sl = sl == PD ? 0 : sl + 1; psl = psl == PD ? 0 : psl + 1;
    }
    WAIT_VM(0);
    for (int j = nf - PD - 1; j < nf; ++j) {
      const int f = wi + j * NPW;
      ++done;
      if (j == nf - 1 || ((f + NPW) >> 7) != (f >> 7)) {
        if (lane == 0) __hip_atomic_fetch_add(cnt + (f >> 7) * 32, (unsigned)done, __ATOMIC_RELAXED, __HIP_MEMORY_SCOPE_SYSTEM);
        done = 0;
      }
    }
    STAMP(32);
    if (tid == 0) __hip_atomic_store(mycnt, call, __ATOMIC_RELAXED, __HIP_MEMORY_SCOPE_AGENT);
    return;
  }

  const int g = bid - NPUSH;
#define TMAP 1
  const int idx = g >> 3;
  const int half = idx & 1, nt = idx >> 1;
  const int src = ((g & 7) + half * 4) & 7;
  const int mt = src * 2 + half;
  const bool has5 = nt < NTN - 1;
  const int wcols = has5 ? 80 : 64;
  const int wm = wv >> 1, wn = wv & 1;
  unsigned xtarget = call * XCONTRIB;
  const unsigned wtarget = call * WCONTRIB;
  unsigned* xcnt = (unsigned*)(ws + OFF_XCNT + (size_t)src * 32 * 128);
  unsigned* wcnt = (unsigned*)(ws + OFF_WCNT) + nt * 32;

  {
    const int kf = mt * 8 + wv;
    const float* wrow = p.w + (size_t)(kf * 32 + (lane >> 4) * 8) * NS + nt * 160;
    char* wdst = ws + OFF_WF + (size_t)kf * WF_KF + (size_t)nt * 10240 + lane * 16;
    f4 a[2][8]; float e[2][8];
    #pragma unroll
    for (int h = 0; h < 2; ++h)
      #pragma unroll
      for (int q = 0; q < 8; ++q) a[h][q] = *(const f4*)(wrow + (size_t)q * NS + h * wcols + (lane & 15) * 4);
    if (has5) {
      #pragma unroll
      for (int h = 0; h < 2; ++h)
        #pragma unroll
        for (int q = 0; q < 8; ++q) e[h][q] = wrow[(size_t)q * NS + h * 80 + 64 + (lane & 15)];
    }
    #pragma unroll
    for (int h = 0; h < 2; ++h) {
      #pragma unroll
      for (int u = 0; u < 4; ++u) {
        h8 v;
        #pragma unroll
        for (int q = 0; q < 8; ++q) v[q] = (_Float16)a[h][q][u];
        st16_sys(wdst + (h * 5 + u) * 1024, __builtin_bit_cast(u4, v));
      }
      if (has5) {
        h8 v;
        #pragma unroll
        for (int q = 0; q < 8; ++q) v[q] = (_Float16)e[h][q];
        st16_sys(wdst + (h * 5 + 4) * 1024, __builtin_bit_cast(u4, v));
      }
    }
    WAIT_VM(0);
    BAR();
    if (tid == 0) __hip_atomic_fetch_add(wcnt, 1u, __ATOMIC_RELAXED, __HIP_MEMORY_SCOPE_AGENT);
  }

  const char* gA = ws + OFF_XF + par * XF_PAR + (size_t)src * XF_RANK +
                   (size_t)((wv >> 2) * 32 + half * 16 + (wv & 3) * 4) * 1024 + lane * 16;
  char* ldA = smem + ((wv >> 2) * 16 + (wv & 3) * 4) * 1024 + lane * 16;
  const int pb2 = wv < 4 ? wv + 16 : wv + 8;
  const char* gB0 = ws + OFF_WF + (size_t)nt * 10240 + lane * 16;
  const size_t ob0 = (size_t)(wv / 10) * WF_KF + (wv % 10) * 1024;
  const size_t ob1 = (size_t)((wv + 8) / 10) * WF_KF + ((wv + 8) % 10) * 1024;
  const size_t ob2 = (size_t)(pb2 / 10) * WF_KF + (pb2 % 10) * 1024;
  char* ldB = smem + 32768 + lane * 16;
  const char* rdA = smem + (wm * 4) * 1024 + lane * 16;
  const char* rdB = smem + 32768 + (wn * 5) * 1024 + lane * 16;

  f4 acc[4][5];
  #pragma unroll
  for (int m = 0; m < 4; ++m)
    #pragma unroll
    for (int n = 0; n < 5; ++n) acc[m][n] = f4{0.f, 0.f, 0.f, 0.f};
  h8 fa[2][4], fb[2][5];
  unsigned fx = 0;

#define ISSUE(S, B) do { \
    _Pragma("unroll") for (int i_ = 0; i_ < 4; ++i_) \
      GLDS(gA + (size_t)(S) * 65536 + i_ * 1024, ldA + (B) * STAGE_B + i_ * 1024); \
    GLDS(gB0 + (size_t)(S) * (2 * WF_KF) + ob0, ldB + (B) * STAGE_B + wv * 1024); \
    GLDS(gB0 + (size_t)(S) * (2 * WF_KF) + ob1, ldB + (B) * STAGE_B + (wv + 8) * 1024); \
    GLDS(gB0 + (size_t)(S) * (2 * WF_KF) + ob2, ldB + (B) * STAGE_B + pb2 * 1024); \
  } while (0)
#define READ(SET, B, KH) do { \
    _Pragma("unroll") for (int n_ = 0; n_ < 5; ++n_) fb[SET][n_] = *(const h8*)(rdB + (B) * STAGE_B + (KH) * 10240 + n_ * 1024); \
    _Pragma("unroll") for (int m_ = 0; m_ < 4; ++m_) fa[SET][m_] = *(const h8*)(rdA + (B) * STAGE_B + (KH) * 16384 + m_ * 1024); \
  } while (0)
#define MMA(SET) do { \
    _Pragma("unroll") for (int m_ = 0; m_ < 4; ++m_) \
      _Pragma("unroll") for (int n_ = 0; n_ < 5; ++n_) \
        acc[m_][n_] = __builtin_amdgcn_mfma_f32_16x16x32_f16(fa[SET][m_], fb[SET][n_], acc[m_][n_], 0, 0, 0); \
  } while (0)
#define FLAGLOAD(Q) asm volatile("global_load_dword %0, %1, off sc0 sc1" : "=v"(fx) : "v"(xcnt + (Q) * 32) : "memory")

  if (wv == 0) {
    spin_wait(wcnt, wtarget, tmo, call);
    spin_wait(xcnt, xtarget, tmo, call);
    spin_wait(xcnt + 32, xtarget, tmo, call);
    __builtin_amdgcn_fence(__ATOMIC_ACQUIRE, "");
    WAIT_VM(0);
  }
  BAR();
  STAMP(1);
  ISSUE(0, 0); ISSUE(1, 1);
  if (wv == 0) FLAGLOAD(2);
  if (wv < 4) ISSUE(2, 2);
  if (wv == 0) { WAIT_VM(15); } else if (wv < 4) { WAIT_VM(14); } else { WAIT_VM(7); }
  BAR();
  READ(0, 0, 0);

#define STEPA(T, P) do { \
    const int t_ = (T); \
    READ(1, (P), 1); \
    MMA(0); \
    if (t_ + 2 >= NSTG) { WAIT_VM(0); } \
    else if (wv == 0 && (t_ & 1) == 0 && t_ + 3 < NSTG - 1) { WAIT_VM(8); } \
    else { WAIT_VM(7); } \
    if (wv == 0 && (t_ & 1) == 1 && t_ + 3 < NSTG) { \
      asm volatile("" : "+v"(fx)); \
      if (__builtin_amdgcn_readfirstlane(fx) < xtarget) spin_wait(xcnt + ((t_ + 3) >> 1) * 32, xtarget, tmo, call); \
      STAMP(1 + ((t_ + 3) >> 1)); \
    } \
    WAIT_LGKM0(); \
    BAR(); \
    if (t_ + 3 < NSTG) ISSUE(t_ + 3, (P)); \
    if (wv == 0 && (t_ & 1) == 1 && t_ + 5 < NSTG) FLAGLOAD((t_ + 5) >> 1); \
    if (t_ + 1 < NSTG) READ(0, ((P) + 1) % 3, 0); \
    MMA(1); \
  } while (0)
#define STEPB(T, P) do { \
    const int t_ = (T); \
    if (t_ + 2 < NSTG) ISSUE(t_ + 2, ((P) + 2) % 3); \
    READ(1, (P), 1); \
    MMA(0); \
    if (t_ + 2 >= NSTG) { WAIT_VM(0); } else { WAIT_VM(7); } \
    WAIT_LGKM0(); \
    BAR(); \
    if (t_ + 1 < NSTG) READ(0, ((P) + 1) % 3, 0); \
    MMA(1); \
  } while (0)

  if (wv < 4) {
    for (int t = 0; t < 63; t += 3) { STEPA(t, 0); STEPA(t + 1, 1); STEPA(t + 2, 2); }
    STEPA(63, 0);
  } else {
    for (int t = 0; t < 63; t += 3) { STEPB(t, 0); STEPB(t + 1, 1); STEPB(t + 2, 2); }
    STEPB(63, 0);
  }

  STAMP(33);
  const bool bad = __hip_atomic_load(tmo, __ATOMIC_RELAXED, __HIP_MEMORY_SCOPE_AGENT) == call;
  const float qnan = __builtin_nanf("");
  float* o = p.out + (size_t)(mt * 256 + wm * 64 + (lane >> 4) * 4) * NS + nt * 160 + wn * wcols;
  #pragma unroll
  for (int m = 0; m < 4; ++m)
    #pragma unroll
    for (int r = 0; r < 4; ++r) {
      f4 v;
      #pragma unroll
      for (int n = 0; n < 4; ++n) v[n] = bad ? qnan : fmaxf(acc[m][n][r], 0.f);
      *(f4*)(o + (size_t)(m * 16 + r) * NS + (lane & 15) * 4) = v;
      if (has5) o[(size_t)(m * 16 + r) * NS + 64 + (lane & 15)] = bad ? qnan : fmaxf(acc[m][4][r], 0.f);
    }
  STAMP(34);
  if (tid == 0) __hip_atomic_store(mycnt, call, __ATOMIC_RELAXED, __HIP_MEMORY_SCOPE_AGENT);
}

}

extern "C" void kernel_launch(int rank, int nranks, ncclComm_t comm,
                              hipStream_t stream,
                              void* const* d_in, const int* in_sizes, int n_in,
                              void* d_out, int out_size,
                              const int* in_dtypes, int out_dtype,
                              void* const* d_out_all,
                              void* const* d_ws_all, size_t ws_bytes) {
  static bool init = false;
  if (!init) {
    if (nranks != NR || ws_bytes < WS_NEED || n_in != 2 || in_sizes[0] != MS * KD ||
        in_sizes[1] != KD * NS || out_size != NR * MS * NS) {
      fprintf(stderr, "unexpected problem shape\n");
      return;
    }
    HIP_CHECK(hipFuncSetAttribute((const void*)ag_gemm_kernel,
                                  hipFuncAttributeMaxDynamicSharedMemorySize, LDS_BYTES));
    init = true;
  }
  Params p;
  p.x = (const float*)d_in[0];
  p.w = (const float*)d_in[1];
  p.out = (float*)d_out;
  p.ws_all = d_ws_all;
  p.rank = rank;
  hipLaunchKernelGGL(ag_gemm_kernel, dim3(NPUSH + NGEMM), dim3(512), LDS_BYTES, stream, p);
}
```

```cpp
#include <hip/hip_runtime.h>
#include <rccl/rccl.h>
#include <cstdio>

#define HIP_CHECK(expr) do { hipError_t _e = (expr); if (_e != hipSuccess) { \
    fprintf(stderr, "HIP error %s at %s:%d\n", hipGetErrorString(_e), __FILE__, __LINE__); \
    return; } } while (0)

namespace {

typedef _Float16 h8 __attribute__((ext_vector_type(8)));
typedef float f4 __attribute__((ext_vector_type(4)));
typedef unsigned u4 __attribute__((ext_vector_type(4)));

constexpr int NR = 8;
constexpr int MS = 512;
constexpr int KD = 4096;
constexpr int NS = 2048;
constexpr int NSTG = KD / 64;
constexpr int NCH = 32;
constexpr int NPUSH = 48, NPW = 48, NTN = 13, NGEMM = 16 * NTN;
constexpr unsigned XCONTRIB = 128;
constexpr unsigned WCONTRIB = 16;

constexpr size_t OFF_WGCNT = 0;
constexpr size_t OFF_TMO = 4096;
constexpr size_t OFF_XCNT = 8192;
constexpr size_t OFF_WCNT = OFF_XCNT + 8 * 32 * 128;
constexpr size_t OFF_XF = 1ull << 20;
constexpr size_t XF_RANK = 4ull << 20;
constexpr size_t XF_PAR = 32ull << 20;
constexpr size_t OFF_WF = OFF_XF + 2 * XF_PAR;
constexpr size_t WF_KF = 130 * 1024;
constexpr size_t WS_NEED = OFF_WF + 128 * WF_KF;

constexpr int STAGE_B = 52 * 1024;
constexpr int LDS_BYTES = 3 * STAGE_B;

constexpr unsigned long long DEADLINE = 50000000ull;

struct Params {
  const float* x;
  const float* w;
  float* out;
  void* const* ws_all;
  int rank;
};

__device__ __forceinline__ void st16_sys(void* p, u4 v) {
  asm volatile("global_store_dwordx4 %0, %1, off sc0 sc1\n\ts_nop 1" :: "v"(p), "v"(v) : "memory");
}

__device__ __forceinline__ u4 cvt8(f4 a, f4 b) {
  h8 h;
  h[0] = (_Float16)a[0]; h[1] = (_Float16)a[1]; h[2] = (_Float16)a[2]; h[3] = (_Float16)a[3];
  h[4] = (_Float16)b[0]; h[5] = (_Float16)b[1]; h[6] = (_Float16)b[2]; h[7] = (_Float16)b[3];
  return __builtin_bit_cast(u4, h);
}

#define STAMP(i) do {} while (0)
#define WAIT_VM(n) asm volatile("s_waitcnt vmcnt(" #n ")" ::: "memory")
#define WAIT_LGKM0() asm volatile("s_waitcnt lgkmcnt(0)" ::: "memory")
#define BAR() __builtin_amdgcn_s_barrier()
#define GLDS(g, l) __builtin_amdgcn_global_load_lds((const unsigned*)(g), (unsigned*)(l), 16, 0, 0)

#define PD 1
__device__ __forceinline__ void wait_vm_n(int n) {
  switch (n) {
    case 0: WAIT_VM(0); break;   case 1: WAIT_VM(1); break;   case 2: WAIT_VM(2); break;
    case 3: WAIT_VM(3); break;   case 4: WAIT_VM(4); break;   case 5: WAIT_VM(5); break;
    case 6: WAIT_VM(6); break;   case 7: WAIT_VM(7); break;   case 8: WAIT_VM(8); break;
    case 9: WAIT_VM(9); break;   case 10: WAIT_VM(10); break; case 11: WAIT_VM(11); break;
    default: WAIT_VM(12); break;
  }
}

__device__ __forceinline__ int frag_cnt(int n, int slot8) {
  int r = n % NPW - slot8; r = r < 0 ? 0 : (r > 8 ? 8 : r);
  return (n / NPW) * 8 + r;
}
__device__ __forceinline__ bool signal_chunk(unsigned* lcnt, unsigned* cnt, int c, int done, int slot8, int lane) {
  unsigned old = 0;
  if (lane == 0) old = __hip_atomic_fetch_add(lcnt + c, (unsigned)done, __ATOMIC_RELAXED, __HIP_MEMORY_SCOPE_WORKGROUP);
  old = __builtin_amdgcn_readfirstlane(old);
  const unsigned expect = frag_cnt(c * 128 + 128, slot8) - frag_cnt(c * 128, slot8);
  if (old + (unsigned)done != expect) return false;
  if (lane == 0) __hip_atomic_fetch_add(cnt + c * 32, expect, __ATOMIC_RELAXED, __HIP_MEMORY_SCOPE_SYSTEM);
  return true;
}

__device__ __forceinline__ void spin_wait(unsigned* flag, unsigned target, unsigned* tmo, unsigned call) {
  unsigned long long t0 = __builtin_amdgcn_s_memrealtime();
  while (__hip_atomic_load(flag, __ATOMIC_RELAXED, __HIP_MEMORY_SCOPE_SYSTEM) < target) {
    if (__hip_atomic_load(tmo, __ATOMIC_RELAXED, __HIP_MEMORY_SCOPE_AGENT) == call ||
        __builtin_amdgcn_s_memrealtime() - t0 > DEADLINE) {
      __hip_atomic_store(tmo, call, __ATOMIC_RELAXED, __HIP_MEMORY_SCOPE_AGENT);
      break;
    }
    __builtin_amdgcn_s_sleep(2);
  }
}

__global__ void __launch_bounds__(512, 2) ag_gemm_kernel(Params p) {
  extern __shared__ __attribute__((aligned(16))) char smem[];
  const int tid = threadIdx.x;
  const int lane = tid & 63;
  const int wv = __builtin_amdgcn_readfirstlane(tid >> 6);
  const int bid = blockIdx.x;
  char* ws = (char*)p.ws_all[p.rank];

#define SRCOFF(F) ((size_t)((F) & 31) * (16 * KD * 4) + (size_t)((F) >> 5) * 128)
  if (bid < NPUSH) {
    const char* src0 = (const char*)(p.x + (size_t)(lane & 15) * KD + (lane >> 4) * 8) + SRCOFF((bid >> 3) * 8 + wv);
    char* ring0 = smem + wv * 8192 + lane * 16;
    GLDS(src0, ring0); GLDS(src0 + 16, ring0 + 1024);
  }
  unsigned* mycnt = (unsigned*)(ws + OFF_WGCNT) + bid;
  const unsigned call = __builtin_amdgcn_readfirstlane(
      __hip_atomic_load(mycnt, __ATOMIC_RELAXED, __HIP_MEMORY_SCOPE_AGENT)) + 1u;
  const unsigned par = call & 1u;
  unsigned* tmo = (unsigned*)(ws + OFF_TMO);
  STAMP(0);

  if (bid < NPUSH) {
    const int role = ((bid & 7) + (bid >> 3)) & 7;
    const int wi = (bid >> 3) * 8 + wv;
    const int dest = (p.rank + 1 + role) & 7;
    char* dws = (char*)p.ws_all[dest];
    char* dst = dws + OFF_XF + par * XF_PAR + (size_t)p.rank * XF_RANK + lane * 16;
    unsigned* cnt = (unsigned*)(dws + OFF_XCNT + (size_t)p.rank * 32 * 128);
    const char* src = (const char*)(p.x + (size_t)(lane & 15) * KD + (lane >> 4) * 8);
    char* ring = smem + wv * 8192 + lane * 16;
    const int nf = (4096 - wi + NPW - 1) / NPW;
    unsigned* lcnt = (unsigned*)(smem + 65536);
    const int slot8 = (bid >> 3) * 8;
    if (tid < NCH) lcnt[tid] = 0;
    __syncthreads();
    #pragma unroll
    for (int i = 1; i < PD; ++i) { const size_t so = SRCOFF(wi + i * NPW); GLDS(src + so, ring + i * 2048); GLDS(src + so + 16, ring + i * 2048 + 1024); }
    int done = 0;
    unsigned hist = 0;
    int sl = 0, psl = PD;
    for (int j = 0; j < nf; ++j) {
      const int f = wi + j * NPW;
      if (j + PD < nf) {
        const size_t so = SRCOFF(f + PD * NPW);
        GLDS(src + so, ring + psl * 2048); GLDS(src + so + 16, ring + psl * 2048 + 1024);
      }
      const int ng = (nf - 1 - j) < PD ? (nf - 1 - j) : PD;
      const int ns = j < PD ? j : PD;
      wait_vm_n(ns + 2 * ng + __builtin_popcount(hist & ((1u << PD) - 1)));
      hist <<= 1;
      if (j >= PD + 1) {
        ++done;
        const int c2 = (f - (PD + 1) * NPW) >> 7;
        if (((f - PD * NPW) >> 7) != c2) {
          if (signal_chunk(lcnt, cnt, c2, done, slot8, lane)) hist |= 1;
          done = 0;
        }
      }
      f4 x0 = *(const f4*)(ring + sl * 2048), x1 = *(const f4*)(ring + sl * 2048 + 1024);
      st16_sys(dst + (size_t)f * 1024, cvt8(x0, x1));
      sl = sl == PD ? 0 : sl + 1; psl = psl == PD ? 0 : psl + 1;
    }
    WAIT_VM(0);
    for (int j = nf - PD - 1; j < nf; ++j) {
      const int f = wi + j * NPW;
      ++done;
      if (j == nf - 1 || ((f + NPW) >> 7) != (f >> 7)) {
        signal_chunk(lcnt, cnt, f >> 7, done, slot8, lane);
        done = 0;
      }
    }
    STAMP(32);
    if (tid == 0) __hip_atomic_store(mycnt, call, __ATOMIC_RELAXED, __HIP_MEMORY_SCOPE_AGENT);
    return;
  }

  const int g = bid - NPUSH;
  const int idx = g >> 3;
  const int half = idx & 1, nt = idx >> 1;
  const int src = ((g & 7) + half * 4) & 7;
  const int mt = src * 2 + half;
  const bool has5 = nt < NTN - 1;
  const int wcols = has5 ? 80 : 64;
  const int wm = wv >> 1, wn = wv & 1;
  unsigned xtarget = call * XCONTRIB;
  const unsigned wtarget = call * WCONTRIB;
  unsigned* xcnt = (unsigned*)(ws + OFF_XCNT + (size_t)src * 32 * 128);
  unsigned* wcnt = (unsigned*)(ws + OFF_WCNT) + nt * 32;

  {
    unsigned warm = 0;
    const int line = tid * NGEMM + g;
    if (line < MS * KD / 32) warm = *(const unsigned*)(p.x + (size_t)line * 32);
    const int kf = mt * 8 + wv;
    const float* wrow = p.w + (size_t)(kf * 32 + (lane >> 4) * 8) * NS + nt * 160;
    char* wdst = ws + OFF_WF + (size_t)kf * WF_KF + (size_t)nt * 10240 + lane * 16;
    f4 a[2][8]; float e[2][8];
    #pragma unroll
    for (int h = 0; h < 2; ++h)
      #pragma unroll
      for (int q = 0; q < 8; ++q) a[h][q] = *(const f4*)(wrow + (size_t)q * NS + h * wcols + (lane & 15) * 4);
    if (has5) {
      #pragma unroll
      for (int h = 0; h < 2; ++h)
        #pragma unroll
        for (int q = 0; q < 8; ++q) e[h][q] = wrow[(size_t)q * NS + h * 80 + 64 + (lane & 15)];
    }
    #pragma unroll
    for (int h = 0; h < 2; ++h) {
      #pragma unroll
      for (int u = 0; u < 4; ++u) {
        h8 v;
        #pragma unroll
        for (int q = 0; q < 8; ++q) v[q] = (_Float16)a[h][q][u];
        st16_sys(wdst + (h * 5 + u) * 1024, __builtin_bit_cast(u4, v));
      }
      if (has5) {
        h8 v;
        #pragma unroll
        for (int q = 0; q < 8; ++q) v[q] = (_Float16)e[h][q];
        st16_sys(wdst + (h * 5 + 4) * 1024, __builtin_bit_cast(u4, v));
      }
    }
    asm volatile("" :: "v"(warm));
    WAIT_VM(0);
    BAR();
    if (tid == 0) __hip_atomic_fetch_add(wcnt, 1u, __ATOMIC_RELAXED, __HIP_MEMORY_SCOPE_AGENT);
  }

  const char* gA = ws + OFF_XF + par * XF_PAR + (size_t)src * XF_RANK +
                   (size_t)((wv >> 2) * 32 + half * 16 + (wv & 3) * 4) * 1024 + lane * 16;
  char* ldA = smem + ((wv >> 2) * 16 + (wv & 3) * 4) * 1024 + lane * 16;
  const int pb2 = wv < 4 ? wv + 16 : wv + 8;
  const char* gB0 = ws + OFF_WF + (size_t)nt * 10240 + lane * 16;
  const size_t ob0 = (size_t)(wv / 10) * WF_KF + (wv % 10) * 1024;
  const size_t ob1 = (size_t)((wv + 8) / 10) * WF_KF + ((wv + 8) % 10) * 1024;
  const size_t ob2 = (size_t)(pb2 / 10) * WF_KF + (pb2 % 10) * 1024;
  char* ldB = smem + 32768 + lane * 16;
  const char* rdA = smem + (wm * 4) * 1024 + lane * 16;
  const char* rdB = smem + 32768 + (wn * 5) * 1024 + lane * 16;

  f4 acc[4][5];
  #pragma unroll
  for (int m = 0; m < 4; ++m)
    #pragma unroll
    for (int n = 0; n < 5; ++n) acc[m][n] = f4{0.f, 0.f, 0.f, 0.f};
  h8 fa[2][4], fb[2][5];
  unsigned fx = 0;

#define ISSUE(S, B) do { \
    _Pragma("unroll") for (int i_ = 0; i_ < 4; ++i_) \
      GLDS(gA + (size_t)(S) * 65536 + i_ * 1024, ldA + (B) * STAGE_B + i_ * 1024); \
    GLDS(gB0 + (size_t)(S) * (2 * WF_KF) + ob0, ldB + (B) * STAGE_B + wv * 1024); \
    GLDS(gB0 + (size_t)(S) * (2 * WF_KF) + ob1, ldB + (B) * STAGE_B + (wv + 8) * 1024); \
    GLDS(gB0 + (size_t)(S) * (2 * WF_KF) + ob2, ldB + (B) * STAGE_B + pb2 * 1024); \
  } while (0)
#define READ(SET, B, KH) do { \
    _Pragma("unroll") for (int n_ = 0; n_ < 5; ++n_) fb[SET][n_] = *(const h8*)(rdB + (B) * STAGE_B + (KH) * 10240 + n_ * 1024); \
    _Pragma("unroll") for (int m_ = 0; m_ < 4; ++m_) fa[SET][m_] = *(const h8*)(rdA + (B) * STAGE_B + (KH) * 16384 + m_ * 1024); \
  } while (0)
#define MMA(SET) do { \
    _Pragma("unroll") for (int m_ = 0; m_ < 4; ++m_) \
      _Pragma("unroll") for (int n_ = 0; n_ < 5; ++n_) \
        acc[m_][n_] = __builtin_amdgcn_mfma_f32_16x16x32_f16(fa[SET][m_], fb[SET][n_], acc[m_][n_], 0, 0, 0); \
  } while (0)
#define FLAGLOAD(Q) asm volatile("v_mov_b32 %0, 0\n\tglobal_load_dword %0, %1, off sc0 sc1" : "=&v"(fx) : "v"(xcnt + (Q) * 32) : "memory")

  if (wv == 0) {
    spin_wait(wcnt, wtarget, tmo, call);
    spin_wait(xcnt, xtarget, tmo, call);
    spin_wait(xcnt + 32, xtarget, tmo, call);
    __builtin_amdgcn_fence(__ATOMIC_ACQUIRE, "");
    WAIT_VM(0);
  }
  BAR();
  STAMP(1);
  ISSUE(0, 0); ISSUE(1, 1);
  if (wv == 0) FLAGLOAD(2);
  if (wv < 4) ISSUE(2, 2);
  if (wv == 0) { WAIT_VM(15); } else if (wv < 4) { WAIT_VM(14); } else { WAIT_VM(7); }
  BAR();
  READ(0, 0, 0);

#define STEPA(T, P) do { \
    const int t_ = (T); \
    READ(1, (P), 1); \
    MMA(0); \
    if (t_ + 2 >= NSTG) { WAIT_VM(0); } \
    else if (wv == 0 && (t_ & 1) == 0 && t_ + 3 < NSTG - 1) { WAIT_VM(8); } \
    else { WAIT_VM(7); } \
    if (wv == 0 && (t_ & 1) == 1 && t_ + 3 < NSTG) { \
      asm volatile("" : "+v"(fx)); \
      if (__builtin_amdgcn_readfirstlane(fx) < xtarget) spin_wait(xcnt + ((t_ + 3) >> 1) * 32, xtarget, tmo, call); \
      STAMP(1 + ((t_ + 3) >> 1)); \
    } \
    WAIT_LGKM0(); \
    BAR(); \
    if (t_ + 3 < NSTG) ISSUE(t_ + 3, (P)); \
    if (wv == 0 && (t_ & 1) == 1 && t_ + 5 < NSTG) FLAGLOAD((t_ + 5) >> 1); \
    if (t_ + 1 < NSTG) READ(0, ((P) + 1) % 3, 0); \
    MMA(1); \
  } while (0)
#define STEPB(T, P) do { \
    const int t_ = (T); \
    if (t_ + 2 < NSTG) ISSUE(t_ + 2, ((P) + 2) % 3); \
    READ(1, (P), 1); \
    MMA(0); \
    if (t_ + 2 >= NSTG) { WAIT_VM(0); } else { WAIT_VM(7); } \
    WAIT_LGKM0(); \
    BAR(); \
    if (t_ + 1 < NSTG) READ(0, ((P) + 1) % 3, 0); \
    MMA(1); \
  } while (0)

  if (wv < 4) {
    for (int t = 0; t < 63; t += 3) { STEPA(t, 0); STEPA(t + 1, 1); STEPA(t + 2, 2); }
    STEPA(63, 0);
  } else {
    for (int t = 0; t < 63; t += 3) { STEPB(t, 0); STEPB(t + 1, 1); STEPB(t + 2, 2); }
    STEPB(63, 0);
  }

  STAMP(33);
  const bool bad = __hip_atomic_load(tmo, __ATOMIC_RELAXED, __HIP_MEMORY_SCOPE_AGENT) == call;
  const float qnan = __builtin_nanf("");
  float* o = p.out + (size_t)(mt * 256 + wm * 64 + (lane >> 4) * 4) * NS + nt * 160 + wn * wcols;
  #pragma unroll
  for (int m = 0; m < 4; ++m)
    #pragma unroll
    for (int r = 0; r < 4; ++r) {
      f4 v;
      #pragma unroll
      for (int n = 0; n < 4; ++n) v[n] = bad ? qnan : fmaxf(acc[m][n][r], 0.f);
      *(f4*)(o + (size_t)(m * 16 + r) * NS + (lane & 15) * 4) = v;
      if (has5) o[(size_t)(m * 16 + r) * NS + 64 + (lane & 15)] = bad ? qnan : fmaxf(acc[m][4][r], 0.f);
    }
  STAMP(34);
  if (tid == 0) __hip_atomic_store(mycnt, call, __ATOMIC_RELAXED, __HIP_MEMORY_SCOPE_AGENT);
}

}

extern "C" void kernel_launch(int rank, int nranks, ncclComm_t comm,
                              hipStream_t stream,
                              void* const* d_in, const int* in_sizes, int n_in,
                              void* d_out, int out_size,
                              const int* in_dtypes, int out_dtype,
                              void* const* d_out_all,
                              void* const* d_ws_all, size_t ws_bytes) {
  static bool init = false;
  if (!init) {
    if (nranks != NR || ws_bytes < WS_NEED || n_in != 2 || in_sizes[0] != MS * KD ||
        in_sizes[1] != KD * NS || out_size != NR * MS * NS) {
      fprintf(stderr, "unexpected problem shape\n");
      return;
    }
    HIP_CHECK(hipFuncSetAttribute((const void*)ag_gemm_kernel,
                                  hipFuncAttributeMaxDynamicSharedMemorySize, LDS_BYTES));
    init = true;
  }
  Params p;
  p.x = (const float*)d_in[0];
  p.w = (const float*)d_in[1];
  p.out = (float*)d_out;
  p.ws_all = d_ws_all;
  p.rank = rank;
  hipLaunchKernelGGL(ag_gemm_kernel, dim3(NPUSH + NGEMM), dim3(512), LDS_BYTES, stream, p);
}
```
